```python
import numpy as np

kernel_name = "gpu_a2a_gemm_m4096_k4096_n4096_e4m3e5m2_dequant"


ABSMAX_RTOL = 0.05

COMPUTE_DTYPE = "float32"
EPILOGUE = "dequant"
IN_QUANT = None
OUT_QUANT = None
IN_FP8 = "e4m3e5m2"
IN_DTYPES = {"x": "e4m3", "w_mat": "e5m2",
             "scale_x": "float32", "scale_w": "float32"}
WORLD_SIZE = 8

def _snap_fp8(x, mant_bits, max_abs, min_normal):
    a = np.abs(x.astype(np.float64))
    _m, _e = np.frexp(a)
    q = np.where(a >= min_normal, np.ldexp(1.0, _e - 1 - mant_bits),
                 min_normal * 2.0 ** -mant_bits)
    snapped = np.minimum(np.round(a / q) * q, max_abs)
    return (np.sign(x) * snapped).astype(np.float32)

def setup_inputs():

    x = _snap_fp8(np.random.randn(4096, 4096).astype(np.float32), 3, 448.0, 2.0 ** -6)
    w_mat = _snap_fp8(np.random.randn(4096, 4096).astype(np.float32), 2, 57344.0, 2.0 ** -14)
    scale_x = (0.02 * (1.0 + np.random.rand(1))).astype(np.float32)
    scale_w = (0.02 * (1.0 + np.random.rand(1))).astype(np.float32)
    return {"x": x, "w_mat": w_mat, "scale_x": scale_x, "scale_w": scale_w}

def reference(x, w_mat, scale_x, scale_w):
    acc = (x @ w_mat).astype(np.float32)
    y = (acc * (scale_x * scale_w)).astype(np.float32)
    return y
```

```cpp
#include <hip/hip_runtime.h>
#include <rccl/rccl.h>
#include <cstdio>
#include <cstdint>

#define HIP_CHECK(expr) do { hipError_t _e = (expr); if (_e != hipSuccess) { \
    fprintf(stderr, "HIP error %s at %s:%d\n", hipGetErrorString(_e), __FILE__, __LINE__); \
    return; } } while (0)

namespace {

typedef int v2i __attribute__((ext_vector_type(2)));
typedef unsigned v4u __attribute__((ext_vector_type(4)));
typedef int v8i __attribute__((ext_vector_type(8)));
typedef float v4f __attribute__((ext_vector_type(4)));
typedef __attribute__((address_space(3))) char lds_char;
typedef __amdgpu_buffer_rsrc_t rsrc_t;

constexpr int NR = 8;
constexpr int KDIM = 4096, NDIM = 4096, MLOC = 512, KLOC = 512;
constexpr unsigned X_BYTES = 4096u * KLOC;
constexpr unsigned W_BYTES = (unsigned)KDIM * NDIM;
constexpr unsigned OUT_BYTES = (unsigned)MLOC * NDIM * 4u;
constexpr unsigned BLK_BYTES = MLOC * KLOC;
constexpr unsigned SLOT_BYTES = NR * BLK_BYTES;
constexpr unsigned NSLOT = 3;
constexpr unsigned OFF_SEQ = 0;
constexpr unsigned OFF_TMO = 16384;
constexpr unsigned OFF_STAGE = 65536;
constexpr unsigned STAGE_BYTES = NSLOT * SLOT_BYTES;
constexpr unsigned WS_USED = OFF_STAGE + STAGE_BYTES;
constexpr unsigned long long DEADLINE_TICKS = 30000000ull;

constexpr int RW = 8, W_SLOT = 16384;
constexpr int RA = 3, A_SLOT = 10240;
constexpr int LDS_A = RW * W_SLOT;
constexpr int LDS_CTL = LDS_A + RA * A_SLOT;
constexpr int LDS_BYTES = LDS_CTL + 128;
constexpr int AUX_SC1 = 16, AUX_SYS = 17;

#define PUSH_WIN 4
#define HINT_BACK 0

struct Params {
  const uint8_t* x; const uint8_t* w; const float* sx; const float* sw; float* out;
  uint8_t* ws[NR];
  int rank;
};

template <int N> __device__ __forceinline__ void waitv() { asm volatile("s_waitcnt vmcnt(%0)" :: "i"(N) : "memory"); }
__device__ __forceinline__ void wait_lgkm0() { asm volatile("s_waitcnt lgkmcnt(0)" ::: "memory"); }
__device__ __forceinline__ unsigned long long rtime() { return __builtin_amdgcn_s_memrealtime(); }
__device__ __forceinline__ rsrc_t mkrsrc(const void* p, unsigned bytes) {
  return __builtin_amdgcn_make_buffer_rsrc((void*)p, (short)0, (int)bytes, 0x00020000);
}
__device__ __forceinline__ int grp_first_piece(int g) { return g < 4 ? 10 * g : 8 * g + 8; }

__device__ __forceinline__ void step_src(int s, int g, int r, int& p, int& j, int& d) {
  if (s < 4) { p = r; j = s; d = 0; }
  else { int i = s - 4; j = (i * 37) >> 8; int im = i - 7 * j; d = g - im; if (d < 0) d += 7; p = (r - 1 - d) & 7; }
}

__device__ __forceinline__ unsigned lds_ld32(lds_char* p) { return *(volatile __attribute__((address_space(3))) unsigned*)p; }
__device__ __forceinline__ void lds_st32(lds_char* p, unsigned v) { *(volatile __attribute__((address_space(3))) unsigned*)p = v; }

template <int MT> struct Frags { v4u a[MT][2]; v2i b[2][4]; };

template <int MT>
__device__ __forceinline__ void mma_step(const Frags<MT>& f, v4f (&acc)[MT][2]) {
  v8i bb[2];
#pragma unroll
  for (int t = 0; t < 2; ++t)
    bb[t] = v8i{f.b[t][0][0], f.b[t][0][1], f.b[t][1][0], f.b[t][1][1], f.b[t][2][0], f.b[t][2][1], f.b[t][3][0], f.b[t][3][1]};
#pragma unroll
  for (int m = 0; m < MT; ++m) {
    v8i aa = v8i{(int)f.a[m][0][0], (int)f.a[m][0][1], (int)f.a[m][0][2], (int)f.a[m][0][3],
                 (int)f.a[m][1][0], (int)f.a[m][1][1], (int)f.a[m][1][2], (int)f.a[m][1][3]};
#pragma unroll
    for (int t = 0; t < 2; ++t)
      acc[m][t] = __builtin_amdgcn_mfma_scale_f32_16x16x128_f8f6f4(aa, bb[t], acc[m][t], 0, 1, 0, 0, 0, 0);
  }
}

template <int MT>
__device__ __forceinline__ void compute_wave(const Params& P, lds_char* lds, int g, int nt, int w, int lane) {
  const rsrc_t rw = mkrsrc(P.w, W_BYTES);
  const int r = P.rank;
  const unsigned wv0 = (unsigned)(32 * w + (lane >> 3)) * NDIM + (unsigned)(((lane & 7) ^ ((lane >> 4) & 3)) << 4) + nt * 128;
  const unsigned wv1 = wv0 ^ 64u;
  const unsigned ldsb = (unsigned)(uintptr_t)lds;
  auto issue_w = [&](int s) {
    int p, j, d; step_src(s, g, r, p, j, d);
    const unsigned so = (unsigned)(p * KLOC + j * 128) * NDIM;
    const unsigned dst = ldsb + (s & (RW - 1)) * W_SLOT + w * 4096;
    asm volatile("s_mov_b32 m0, %0\n\ts_nop 0\n\t"
                 "buffer_load_dwordx4 %1, %3, %4 offen lds\n\t"
                 "buffer_load_dwordx4 %1, %3, %5 offen offset:1024 lds\n\t"
                 "buffer_load_dwordx4 %2, %3, %6 offen offset:2048 lds\n\t"
                 "buffer_load_dwordx4 %2, %3, %7 offen offset:3072 lds"
                 :: "s"(dst), "v"(wv0), "v"(wv1), "s"(rw), "s"(so), "s"(so + (8 * NDIM - 1024)),
                    "s"(so + 2 * (8 * NDIM - 1024)), "s"(so + 3 * (8 * NDIM - 1024)) : "memory");
  };
#pragma unroll
  for (int s = 0; s < RW; ++s) issue_w(s);
  __builtin_amdgcn_s_barrier();

  const int gq = lane >> 4, l15 = lane & 15;
  const int q = l15 >> 1;
  const int fsw = ((q >> 1) & 3) | ((gq & 1) << 2);
  const int boff0 = (16 * gq + q) * 128 + (((2 * w) ^ fsw) << 4) + 8 * (lane & 1);
  const int boff1 = boff0 ^ 16;
  const int aoff0 = l15 * 128 + ((gq ^ (l15 & 7)) << 4);
  const int aoff1 = aoff0 ^ 64;
  lds_char* ctl = lds + LDS_CTL;
  lds_char* ctl_rd = ctl + l15 * 4;

  v4f acc[MT][2];
#pragma unroll
  for (int m = 0; m < MT; ++m) { acc[m][0] = v4f{0, 0, 0, 0}; acc[m][1] = v4f{0, 0, 0, 0}; }

  waitv<4 * (RW - 2)>();
  lds_st32(ctl + 16 + w * 4, 2);

  auto read_frags = [&](int s, int aslot, Frags<MT>& f) {
    lds_char* wb = lds + (s & (RW - 1)) * W_SLOT;
    lds_char* ab = lds + LDS_A + aslot * A_SLOT;
#pragma unroll
    for (int u = 0; u < 4; ++u) {
      const int o = (u >> 1) * 8192 + (u & 1) * 1024;
      f.b[0][u] = __builtin_amdgcn_ds_read_tr8_b64_v2i32((__attribute__((address_space(3))) v2i*)(wb + boff0 + o));
      f.b[1][u] = __builtin_amdgcn_ds_read_tr8_b64_v2i32((__attribute__((address_space(3))) v2i*)(wb + boff1 + o));
    }
#pragma unroll
    for (int m = 0; m < MT; ++m) {
      f.a[m][0] = *(__attribute__((address_space(3))) v4u*)(ab + aoff0 + m * 2048);
      f.a[m][1] = *(__attribute__((address_space(3))) v4u*)(ab + aoff1 + m * 2048);
    }
  };
  auto wait_ready = [&](int s) {
    while (true) {
      unsigned v = lds_ld32(ctl_rd);
      if (__all(v > (unsigned)s)) break;
    }
    asm volatile("" ::: "memory");
  };
  auto post = [&](int s) {
    asm volatile("" ::: "memory");
    lds_st32(ctl + 32 + w * 4, s + 3);
    if (s >= 2 && s + RW - 2 < 32) issue_w(s + RW - 2);
  };
  auto wait_w = [&](int s) {
    if (s < 26) waitv<16>(); else if (s == 26) waitv<12>(); else if (s == 27) waitv<8>(); else if (s == 28) waitv<4>(); else waitv<0>();
    lds_st32(ctl + 16 + w * 4, s + 3);
  };

  Frags<MT> f0, f1;
  int aslot = 0;
  wait_ready(0);
  read_frags(0, 0, f0); post(0); wait_w(0);
  aslot = 1;
  for (int s = 1; s < 31; s += 2) {
    wait_ready(s);
    read_frags(s, aslot, f1); post(s);
    aslot = aslot == RA - 1 ? 0 : aslot + 1;
    mma_step<MT>(f0, acc);
    wait_w(s);
    wait_ready(s + 1);
    read_frags(s + 1, aslot, f0); post(s + 1);
    aslot = aslot == RA - 1 ? 0 : aslot + 1;
    mma_step<MT>(f1, acc);
    wait_w(s + 1);
  }
  wait_ready(31);
  read_frags(31, aslot, f1);
  mma_step<MT>(f0, acc);
  mma_step<MT>(f1, acc);

  float scale = P.sx[0] * P.sw[0];
  if (lds_ld32(ctl + 64) != 0) scale = __builtin_nanf("");
  const rsrc_t ro = mkrsrc(P.out, OUT_BYTES);
  const unsigned ov = (unsigned)(grp_first_piece(g) * 8 + 4 * gq) * (NDIM * 4) + (unsigned)(nt * 128 + 32 * w + l15) * 4;
#pragma unroll
  for (int m = 0; m < MT; ++m)
#pragma unroll
    for (int j = 0; j < 4; ++j)
#pragma unroll
      for (int t = 0; t < 2; ++t)
        __builtin_amdgcn_raw_buffer_store_b32(__builtin_bit_cast(unsigned, acc[m][t][j] * scale), ro, ov + t * 64,
                                              (m * 16 + j) * (NDIM * 4), AUX_SC1);
}

template <int MT>
__device__ __forceinline__ void loader_wave(const Params& P, lds_char* lds, int g, int v, int lane, int b) {
  constexpr int NP = 2 * MT;
  const int r = P.rank;
  uint8_t* me = P.ws[r];
  unsigned* seqp = (unsigned*)(me + OFF_SEQ) + b * 8 + 4 + v;
  const unsigned seq = __hip_atomic_load(seqp, __ATOMIC_RELAXED, __HIP_MEMORY_SCOPE_AGENT);
  const rsrc_t rx = mkrsrc(P.x, X_BYTES);
  const rsrc_t rs = mkrsrc(me + OFF_STAGE, STAGE_BYTES);
  const int p0 = grp_first_piece(g);
  v4u a[NP];
  {
    const unsigned vo = (unsigned)(lane >> 3) * KLOC + (lane & 7) * 16;
    const unsigned so = (unsigned)(r * MLOC + p0 * 8) * KLOC + v * 128;
#pragma unroll
    for (int i = 0; i < NP; ++i) a[i] = __builtin_amdgcn_raw_buffer_load_b128(rx, vo, so + i * 8 * KLOC, 0);
  }
  lds_char* ctl = lds + LDS_CTL;
  if (v == 0 && lane < 32) {
    unsigned init = lane < 4 ? lane : lane < 8 ? 0 : lane < 12 ? 2 : lane < 16 ? 0xffffffffu : 0;
    lds_st32(ctl + lane * 4, init);
    wait_lgkm0();
  }
  __builtin_amdgcn_s_barrier();
  const unsigned call = __builtin_amdgcn_readfirstlane(seq) + 1;
  if (lane == 0) __hip_atomic_store(seqp, call, __ATOMIC_RELAXED, __HIP_MEMORY_SCOPE_AGENT);
  const unsigned slot_off = (call % NSLOT) * SLOT_BYTES;
  const int woff = (lane >> 3) * 128 + (((lane & 7) ^ (lane >> 3)) << 4);
  lds_char* done_rd = ctl + 32 + (lane & 3) * 4;
  bool failed = false;
  unsigned hint = 0;
  auto unit_off = [&](int s, unsigned& hint_off) {
    int p, j, d; step_src(s, g, r, p, j, d);
    const unsigned blk = slot_off + p * BLK_BYTES;
    int n = j * 64 + ((p0 + NP - 1 - grp_first_piece(d)) & 63) - HINT_BACK;
    if (n < 0) n = 0;
    hint_off = blk + (n >> 6) * 65536 + ((grp_first_piece(d) + n) & 63) * 1024 + 1020;
    return blk + j * 65536 + p0 * 1024;
  };
  for (int s = v; s < 32; s += 4) {
    if (s >= 4) {
      unsigned ho;
      const unsigned uo = unit_off(s, ho);
      const unsigned long long t0 = rtime();
      while (hint == 0 && !failed) {
        hint = __builtin_amdgcn_readfirstlane(__builtin_amdgcn_raw_buffer_load_b32(rs, 0, ho, AUX_SYS));
        if (hint == 0 && rtime() - t0 > DEADLINE_TICKS) failed = true;
      }
      unsigned hn = 0;
      unsigned ho2 = ho;
      if (s + 4 < 32) unit_off(s + 4, ho2);
      while (true) {
#pragma unroll
        for (int i = 0; i < NP; ++i) a[i] = __builtin_amdgcn_raw_buffer_load_b128(rs, lane * 16, uo + i * 1024, AUX_SYS);
        hn = __builtin_amdgcn_raw_buffer_load_b32(rs, 0, ho2, AUX_SYS);
        unsigned mn = 0xffffffffu;
#pragma unroll
        for (int i = 0; i < NP; ++i) mn = min(min(mn, min(a[i][0], a[i][1])), min(a[i][2], a[i][3]));
        if (__all(mn != 0) || failed) break;
        if (rtime() - t0 > DEADLINE_TICKS) failed = true;
      }
      hint = __builtin_amdgcn_readfirstlane(hn);
    }
    if (s >= RA) {
      while (true) {
        unsigned dn = lds_ld32(done_rd);
        if (__all(dn >= (unsigned)(s - RA + 3))) break;
      }
      asm volatile("" ::: "memory");
    }
    lds_char* ab = lds + LDS_A + (s % RA) * A_SLOT + woff;
#pragma unroll
    for (int i = 0; i < NP; ++i) *(__attribute__((address_space(3))) v4u*)(ab + i * 1024) = a[i];
    asm volatile("" ::: "memory");
    if (failed) lds_st32(ctl + 64, 1);
    lds_st32(ctl + v * 4, s + 4);
  }
  if (failed && lane == 0) __hip_atomic_store((unsigned*)(me + OFF_TMO), call, __ATOMIC_RELAXED, __HIP_MEMORY_SCOPE_AGENT);
  __builtin_amdgcn_s_waitcnt(0x0F70);
}

__device__ __forceinline__ void comm_block(const Params& P, int b, int wave, int lane) {
  const int r = P.rank;
  uint8_t* me = P.ws[r];
  unsigned* seqp = (unsigned*)(me + OFF_SEQ) + b * 8 + wave;
  if (b < 28) {
    if (wave >= 4) return;
    const unsigned seq = __hip_atomic_load(seqp, __ATOMIC_RELAXED, __HIP_MEMORY_SCOPE_AGENT);
    const int d = b % 7, q = b / 7;
    const unsigned dst = (unsigned)(r + 1 + d) & 7u;
    const rsrc_t rx = mkrsrc(P.x, X_BYTES);
    const int st = grp_first_piece(d);
    const unsigned vo = (unsigned)(lane >> 3) * KLOC + (lane & 7) * 16;
    v4u a[16];
#pragma unroll
    for (int m = 0; m < 16; ++m) {
      const int n = q + 4 * wave + 16 * m;
      const int j = n >> 6, rp = (st + n) & 63;
      a[m] = __builtin_amdgcn_raw_buffer_load_b128(rx, vo, (dst * MLOC + rp * 8) * KLOC + j * 128, 0);
    }
    const unsigned call = __builtin_amdgcn_readfirstlane(seq) + 1;
    const rsrc_t rd = mkrsrc(P.ws[dst] + OFF_STAGE + (call % NSLOT) * SLOT_BYTES + r * BLK_BYTES, BLK_BYTES);
#pragma unroll
    for (int m = 0; m < 16; ++m) {
      const int n = q + 4 * wave + 16 * m;
      const int j = n >> 6, rp = (st + n) & 63;
      v4u t = a[m];
#pragma unroll
      for (int c = 0; c < 4; ++c) t[c] = t[c] == 0 ? 0x80u : t[c];
      if (m >= PUSH_WIN) waitv<PUSH_WIN - 1>();
      __builtin_amdgcn_raw_buffer_store_b128(t, rd, lane * 16, j * 65536 + rp * 1024, AUX_SYS);
    }
    if (lane == 0) __hip_atomic_store(seqp, call, __ATOMIC_RELAXED, __HIP_MEMORY_SCOPE_AGENT);
  } else {
    const unsigned seq = __hip_atomic_load(seqp, __ATOMIC_RELAXED, __HIP_MEMORY_SCOPE_AGENT);
    const unsigned call = __builtin_amdgcn_readfirstlane(seq) + 1;
    const rsrc_t rz = mkrsrc(me + OFF_STAGE + ((call + 2) % NSLOT) * SLOT_BYTES, SLOT_BYTES);
    const v4u z = {0, 0, 0, 0};
    const int u = (b - 28) * 8 + wave;
    for (int i = 0; i < 56; ++i) {
      unsigned pc = u + 32 * i;
      if (pc >= (unsigned)r * 256) pc += 256;
      __builtin_amdgcn_raw_buffer_store_b128(z, rz, lane * 16, pc * 1024, AUX_SYS);
    }
    if (lane == 0) __hip_atomic_store(seqp, call, __ATOMIC_RELAXED, __HIP_MEMORY_SCOPE_AGENT);
  }
}

__global__ void __launch_bounds__(512, 2) a2a_gemm(Params P) {
  extern __shared__ __attribute__((aligned(16))) char smem[];
  lds_char* lds = (lds_char*)smem;
  const int b = blockIdx.x, lane = threadIdx.x & 63;
  const int wave = __builtin_amdgcn_readfirstlane(threadIdx.x >> 6);
  if (b < 32) {
    comm_block(P, b, wave, lane);
  } else {
    const int cb = b - 32, g = cb >> 5, nt = (cb & 7) * 4 + ((cb >> 3) & 3);
    if (g < 4) {
      if (wave < 4) compute_wave<5>(P, lds, g, nt, wave, lane); else loader_wave<5>(P, lds, g, wave - 4, lane, b);
    } else {
      if (wave < 4) compute_wave<4>(P, lds, g, nt, wave, lane); else loader_wave<4>(P, lds, g, wave - 4, lane, b);
    }
  }
}

}

extern "C" void kernel_launch(int rank, int nranks, ncclComm_t comm, hipStream_t stream,
                              void* const* d_in, const int* in_sizes, int n_in,
                              void* d_out, int out_size, const int* in_dtypes, int out_dtype,
                              void* const* d_out_all, void* const* d_ws_all, size_t ws_bytes) {
  if (nranks != NR || n_in < 4 || ws_bytes < WS_USED || (unsigned)rank >= (unsigned)NR ||
      in_sizes[0] != (int)X_BYTES || in_sizes[1] != (int)W_BYTES || out_size != MLOC * NDIM) {
    fprintf(stderr, "unexpected problem shape\n");
    return;
  }
  HIP_CHECK(hipFuncSetAttribute((const void*)a2a_gemm, hipFuncAttributeMaxDynamicSharedMemorySize, LDS_BYTES));
  Params P;
  P.x = (const uint8_t*)d_in[0]; P.w = (const uint8_t*)d_in[1];
  P.sx = (const float*)d_in[2]; P.sw = (const float*)d_in[3];
  P.out = (float*)d_out;
  for (int p = 0; p < NR; ++p) P.ws[p] = (uint8_t*)d_ws_all[p];
  P.rank = rank;
  hipLaunchKernelGGL(a2a_gemm, dim3(256), dim3(512), LDS_BYTES, stream, P);
  HIP_CHECK(hipGetLastError());
}
```

```cpp
#include <hip/hip_runtime.h>
#include <rccl/rccl.h>
#include <cstdio>
#include <cstdint>

#define HIP_CHECK(expr) do { hipError_t _e = (expr); if (_e != hipSuccess) { \
    fprintf(stderr, "HIP error %s at %s:%d\n", hipGetErrorString(_e), __FILE__, __LINE__); \
    return; } } while (0)

namespace {

typedef int v2i __attribute__((ext_vector_type(2)));
typedef unsigned v4u __attribute__((ext_vector_type(4)));
typedef int v8i __attribute__((ext_vector_type(8)));
typedef float v4f __attribute__((ext_vector_type(4)));
typedef __attribute__((address_space(3))) char lds_char;
typedef __amdgpu_buffer_rsrc_t rsrc_t;

constexpr int NWAVE = 12;
constexpr int NR = 8;
constexpr int KDIM = 4096, NDIM = 4096, MLOC = 512, KLOC = 512;
constexpr unsigned X_BYTES = 4096u * KLOC;
constexpr unsigned W_BYTES = (unsigned)KDIM * NDIM;
constexpr unsigned OUT_BYTES = (unsigned)MLOC * NDIM * 4u;
constexpr unsigned BLK_BYTES = MLOC * KLOC;
constexpr unsigned SLOT_BYTES = NR * BLK_BYTES;
constexpr unsigned NSLOT = 3;
constexpr unsigned OFF_SEQ = 0;
constexpr unsigned OFF_TMO = 16384;
constexpr unsigned OFF_STAGE = 65536;
constexpr unsigned STAGE_BYTES = NSLOT * SLOT_BYTES;
constexpr unsigned WS_USED = OFF_STAGE + STAGE_BYTES;
constexpr unsigned long long DEADLINE_TICKS = 30000000ull;

constexpr int RING = 6;
constexpr int W_BYTES_STEP = 16384, A_BYTES_STEP = 10240, SLOT = W_BYTES_STEP + A_BYTES_STEP;
constexpr int LDS_CTL = RING * SLOT;
constexpr int LDS_BYTES = LDS_CTL + 64;
constexpr int AUX_SC1 = 16, AUX_SYS = 17;

#define PROF_GAP 2000000u
#define PROF_EVERY 0
#define PROF_S 7
#define LOAD_PRIO 1
#define POLL_WIN 6
#define PUSH_WIN 4
#define HINT_BACK 0

struct Params {
  const uint8_t* x; const uint8_t* w; const float* sx; const float* sw; float* out;
  uint8_t* me;
  uint8_t* peer[NR];
  int rank;
};

#define STAMP(i)
#define TSARG
#define TSPASS
template <int N> __device__ __forceinline__ void waitv() { asm volatile("s_waitcnt vmcnt(%0)" :: "i"(N) : "memory"); }
__device__ __forceinline__ void wait_lgkm0() { asm volatile("" ::: "memory"); __builtin_amdgcn_s_waitcnt(0xC07F); asm volatile("" ::: "memory"); }
__device__ __forceinline__ unsigned long long rtime() { return __builtin_amdgcn_s_memrealtime(); }
__device__ __forceinline__ rsrc_t mkrsrc(const void* p, unsigned bytes) {
  return __builtin_amdgcn_make_buffer_rsrc((void*)p, (short)0, (int)bytes, 0x00020000);
}
__device__ __forceinline__ int grp_first_piece(int g) { return 8 * g + 2 * min(g, 4); }

__device__ __forceinline__ void step_src(int s, int g, int r, int& p, int& j, int& d) {
  const int i = s - 4, jr = (i * 37) >> 8, dr = g - (i - 7 * jr);
  d = dr < 0 ? dr + 7 : dr;
  j = s < 4 ? s : jr;
  p = s < 4 ? r : (r - 1 - d) & 7;
}

__device__ __forceinline__ unsigned lds_ld32(lds_char* p) { return *(volatile __attribute__((address_space(3))) unsigned*)p; }
__device__ __forceinline__ void lds_st32(lds_char* p, unsigned v) { *(volatile __attribute__((address_space(3))) unsigned*)p = v; }

#define SCHED0() __builtin_amdgcn_sched_barrier(0)
#define BARRIER() do { asm volatile("" ::: "memory"); __builtin_amdgcn_s_barrier(); asm volatile("" ::: "memory"); } while (0)
template <int MT> struct Frags { v4u a[MT][2]; v8i b[2]; };

__device__ __forceinline__ void mma1(const v4u (&a)[2], const v8i& bb, v4f& acc) {
  const v8i aa = v8i{(int)a[0][0], (int)a[0][1], (int)a[0][2], (int)a[0][3], (int)a[1][0], (int)a[1][1], (int)a[1][2], (int)a[1][3]};
  acc = __builtin_amdgcn_mfma_scale_f32_16x16x128_f8f6f4(aa, bb, acc, 0, 1, 0, 0, 0, 0);
  asm volatile("" : "+v"(acc));
}

template <int MT>
__device__ __forceinline__ void compute_wave(const Params& P, lds_char* lds, int g, int nt, int w, int lane TSARG) {
  const int gq = lane >> 4, l15 = lane & 15;
  const int q = l15 >> 1;
  const int fsw = ((q >> 1) & 3) | ((gq & 1) << 2);
  const int boff0 = (16 * gq + q) * 128 + (((2 * w) ^ fsw) << 4) + 8 * (lane & 1);
  const int aoff0 = W_BYTES_STEP + l15 * 128 + ((gq ^ (l15 & 7)) << 4);
  lds_char* ctl = lds + LDS_CTL;

  v4f acc[MT][2];
#pragma unroll
  for (int m = 0; m < MT; ++m) { acc[m][0] = v4f{0, 0, 0, 0}; acc[m][1] = v4f{0, 0, 0, 0}; }

  int ob0 = boff0, ob1 = boff0 ^ 16, oa0 = aoff0, oa1 = aoff0 ^ 64;
  asm volatile("" : "+v"(ob0), "+v"(ob1), "+v"(oa0), "+v"(oa1));
  lds_char* pb[2]; lds_char* pa[2];
  int so = 0;
  auto set_addr = [&]() {
    pb[0] = lds + (ob0 + so); pb[1] = lds + (ob1 + so);
    pa[0] = lds + (oa0 + so); pa[1] = lds + (oa1 + so);
    asm volatile("" : "+v"(pb[0]), "+v"(pb[1]), "+v"(pa[0]), "+v"(pa[1]));
  };
  auto read_b = [&](Frags<MT>& f, int t) {
    v2i r[4];
#pragma unroll
    for (int u = 0; u < 4; ++u)
      r[u] = __builtin_amdgcn_ds_read_tr8_b64_v2i32((__attribute__((address_space(3))) v2i*)(pb[t] + (u >> 1) * 8192 + (u & 1) * 1024));
    f.b[t] = v8i{r[0][0], r[0][1], r[1][0], r[1][1], r[2][0], r[2][1], r[3][0], r[3][1]};
  };
  auto read_a = [&](Frags<MT>& f, int m) {
    f.a[m][0] = *(__attribute__((address_space(3))) v4u*)(pa[0] + m * 2048);
    f.a[m][1] = *(__attribute__((address_space(3))) v4u*)(pa[1] + m * 2048);
  };
  auto step = [&](Frags<MT>& fn, const Frags<MT>& fp) {
#pragma unroll
    for (int i = 0; i < 2 * MT; ++i) {
      mma1(fp.a[i >> 1], fp.b[i & 1], acc[i >> 1][i & 1]);
      if (i == 0) { so = so == (RING - 1) * SLOT ? 0 : so + SLOT; set_addr(); }
      if (i == 1) { wait_lgkm0(); BARRIER(); read_a(fn, 0); }
      if (i == 2) read_b(fn, 0);
      if (i == 3) read_b(fn, 1);
      if (i >= 4 && i < MT + 3) read_a(fn, i - 3);
      SCHED0();
    }
  };

  Frags<MT> f0, f1;
  set_addr();
  BARRIER();
  STAMP(2);
  read_a(f0, 0); read_b(f0, 0); read_b(f0, 1);
#pragma unroll
  for (int m = 1; m < MT; ++m) read_a(f0, m);
  SCHED0();
  for (int s = 1; s < 31; s += 2) {
    step(f1, f0);
    step(f0, f1);
  }
  step(f1, f0);
  STAMP(10);
  const unsigned fail = lds_ld32(ctl + 4);
  const float scale0 = P.sx[0] * P.sw[0];
#pragma unroll
  for (int i = 0; i < 2 * MT; ++i) mma1(f1.a[i >> 1], f1.b[i & 1], acc[i >> 1][i & 1]);

  const float scale = fail != 0 ? __builtin_nanf("") : scale0;
  STAMP(11);
  const rsrc_t ro = mkrsrc(P.out, OUT_BYTES);
  const unsigned ov = (unsigned)(grp_first_piece(g) * 8 + 4 * gq) * (NDIM * 4) + (unsigned)(nt * 128 + 32 * w + l15) * 4;
#pragma unroll
  for (int m = 0; m < MT; ++m)
#pragma unroll
    for (int j = 0; j < 4; ++j)
#pragma unroll
      for (int t = 0; t < 2; ++t)
        __builtin_amdgcn_raw_buffer_store_b32(__builtin_bit_cast(unsigned, acc[m][t][j] * scale), ro, ov + t * 64,
                                              (m * 16 + j) * (NDIM * 4), AUX_SC1);
}

template <int PER> __device__ __forceinline__ void wait_steps(int n) {
  waitv<4 * PER>();
  if (n < 4) { waitv<3 * PER>(); if (n < 3) { waitv<2 * PER>(); if (n < 2) { waitv<PER>(); if (n < 1) waitv<0>(); } } }
}

#define DMA4(POL, dst, v0, v1, rs, s0, s1, s2, s3) \
  asm volatile("s_mov_b32 m0, %0\n\ts_nop 3\n\t" \
               "buffer_load_dwordx4 %1, %3, %4 offen" POL " lds\n\t" \
               "buffer_load_dwordx4 %1, %3, %5 offen offset:1024" POL " lds\n\t" \
               "buffer_load_dwordx4 %2, %3, %6 offen offset:2048" POL " lds\n\t" \
               "buffer_load_dwordx4 %2, %3, %7 offen offset:3072" POL " lds" \
               :: "s"(dst), "v"(v0), "v"(v1), "s"(rs), "s"(s0), "s"(s1), "s"(s2), "s"(s3) : "memory")

__device__ __forceinline__ void wload_wave(const Params& P, lds_char* lds, int g, int nt, int u, int lane TSARG) {
  const rsrc_t rw = mkrsrc(P.w, W_BYTES);
  const int r = P.rank;
  const unsigned wv0 = (unsigned)(32 * u + (lane >> 3)) * NDIM + (unsigned)(((lane & 7) ^ ((lane >> 4) & 3)) << 4) + nt * 128;
  const unsigned wv1 = wv0 ^ 64u;
  const unsigned ldsb = (unsigned)(uintptr_t)lds + u * 4096;
  unsigned so = 0;
  unsigned koff;
  { int p, j, d; step_src(lane & 31, g, r, p, j, d); koff = (unsigned)(p * KLOC + j * 128) * NDIM; }
  constexpr unsigned ST = 8 * NDIM - 1024;
  unsigned o0, o1, o2, o3, dst;
  auto form = [&](int s) {
    o0 = s < 4 ? (unsigned)(r * KLOC + s * 128) * NDIM : __builtin_amdgcn_readlane(koff, s);
    o1 = o0 + ST; o2 = o0 + 2 * ST; o3 = o0 + 3 * ST;
    dst = ldsb + so;
    so = so == (RING - 1) * SLOT ? 0 : so + SLOT;
    asm volatile("" : "+s"(o0), "+s"(o1), "+s"(o2), "+s"(o3), "+s"(dst), "+s"(so));
  };
  auto issue_w = [&](int next) {
    DMA4("", dst, wv0, wv1, rw, o0, o1, o2, o3);
    form(next);
  };
  __builtin_amdgcn_s_setprio(LOAD_PRIO);
  form(0);
#pragma unroll
  for (int s = 0; s < RING - 1; ++s) issue_w(s + 1);
  waitv<4 * (RING - 2)>();
  STAMP(1);
  for (int s = 0; s < 32 - (RING - 1); ++s) {
    BARRIER();
    issue_w((s + RING) & 31);
    waitv<4 * (RING - 2)>();
  }
  BARRIER(); waitv<12>();
  BARRIER(); waitv<8>();
  BARRIER(); waitv<4>();
  BARRIER(); waitv<0>();
  BARRIER();
  STAMP(2);
}

#define DMA1(POL, dst, v0, rs, s0) \
  asm volatile("s_mov_b32 m0, %0\n\ts_nop 3\n\t" \
               "buffer_load_dwordx4 %1, %2, %3 offen" POL " lds" \
               :: "s"(dst), "v"(v0), "s"(rs), "s"(s0) : "memory")

template <int MT>
__device__ __forceinline__ void aload_wave(const Params& P, lds_char* lds, int g, int aw, int lane, int b TSARG) {
  constexpr int NP = MT;
  const int role = aw >> 1, half = aw & 1;
  const int r = P.rank;
  uint8_t* me = P.me;
  const rsrc_t rs = mkrsrc(me + OFF_STAGE, STAGE_BYTES);
  const int p0 = grp_first_piece(g) + half * MT;
  unsigned* seqp = (unsigned*)(me + OFF_SEQ) + b * NWAVE + 4 + aw;
  unsigned seqv;
  asm volatile("global_load_dword %0, %1, %2 sc1" : "=v"(seqv) : "v"(0), "s"(seqp) : "memory");
  const unsigned sw = (unsigned)(((lane & 7) ^ (lane >> 3)) << 4);
  const unsigned ldsb = (unsigned)(uintptr_t)lds + W_BYTES_STEP + half * MT * 1024;
  lds_char* ctl = lds + LDS_CTL;
  __builtin_amdgcn_s_setprio(LOAD_PRIO);
  if (role == 0) {
    const rsrc_t rx = mkrsrc(P.x, X_BYTES);
    const unsigned vx = (unsigned)(lane >> 3) * KLOC + sw;
    constexpr unsigned ST = 8 * KLOC - 1024;
#pragma unroll
    for (int s = 0; s < 4; ++s) {
      const unsigned mo = (unsigned)(r * MLOC + p0 * 8) * KLOC + s * 128, dst = ldsb + s * SLOT;
      DMA4("", dst, vx, vx, rx, mo, mo + ST, mo + 2 * ST, mo + 3 * ST);
      if (MT == 5) DMA1("", dst + 4096, vx, rx, mo + 32 * KLOC);
    }
  } else {
    if (half == 0 && lane == 0) lds_st32(ctl + 4, 0);
    wait_lgkm0();
  }
  asm volatile("s_waitcnt vmcnt(%1)" : "+v"(seqv) : "i"(4 * NP) : "memory");
  if (role == 1) asm volatile("s_waitcnt vmcnt(0)" : "+v"(seqv) :: "memory");
  const unsigned call = __builtin_amdgcn_readfirstlane(seqv) + 1;
  const unsigned slot_off = (call % NSLOT) * SLOT_BYTES;
  unsigned hoff, uoff;
  {
    int p, j, d; step_src(min(lane, 27) + 4, g, r, p, j, d);
    const int fd = grp_first_piece(d);
    const int n = max(j * 64 + ((p0 + NP - 1 - fd) & 63) - HINT_BACK, 0);
    hoff = slot_off + p * BLK_BYTES + (n >> 6) * 65536 + ((fd + n) & 63) * 1024 + 1020;
    uoff = slot_off + p * BLK_BYTES + j * 65536 + p0 * 1024;
  }
  const unsigned va = (unsigned)(lane >> 3) * 128 + sw;
  bool failed = false;
  if (role == 0) {
    asm volatile("v_mov_b32 v167, 0" ::: "v167");
    auto poll = [&](int first) {
      const unsigned long long m = (((1ull << POLL_WIN) - 1) << first) & 0xfffffffull;
      asm volatile("s_mov_b64 exec, %0\n\tbuffer_load_dword v167, %1, %2, 0 offen sc0 sc1\n\ts_mov_b64 exec, -1"
                   :: "s"(m), "v"(hoff), "s"(rs) : "memory", "v167");
    };
    auto arrived = [&](int first) {
      unsigned hv;
      asm volatile("v_mov_b32 %0, v167" : "=v"(hv) :: "v167");
      const unsigned m = (unsigned)__ballot(hv != 0);
      return __builtin_ctz(~(m >> first));
    };
    int issued = 4, known = 0, s = 0;
    unsigned so = 4 * SLOT, mo, dst;
    auto form = [&]() {
      mo = __builtin_amdgcn_readlane(uoff, (issued - 4) & 31);
      dst = ldsb + so;
      so = so == (RING - 1) * SLOT ? 0 : so + SLOT;
      asm volatile("" : "+s"(mo), "+s"(dst), "+s"(so));
    };
    auto issue_a = [&]() {
      form();
      ++issued;
      unsigned m1 = mo + 4096, d1 = dst + 4096;
      asm volatile("" : "+s"(m1), "+s"(d1), "+s"(issued));
      DMA4(" sc0 sc1", dst, va, va, rs, mo, mo, mo, mo);
      if (MT == 5) DMA1(" sc0 sc1", d1, va, rs, m1);
    };
    poll(0);
    waitv<2 * NP>();
    STAMP(1);
    while (issued < 32) {
      BARRIER();
      known = arrived(issued - 4);
      if (known == 0 && issued <= s + 2) {
        const unsigned long long t0 = rtime();
        while (!failed) {
          poll(issued - 4);
          waitv<0>();
          known = arrived(issued - 4);
          if (known) break;
          if (rtime() - t0 > DEADLINE_TICKS) failed = true;
        }
        if (failed) { lds_st32(ctl + 4, 1); wait_lgkm0(); known = 1; }
      }
      const int k = min(known, min(s + RING, 32) - issued);
      for (int i = 0; i < k; ++i) issue_a();
      known -= k;
      if (known < 4 && issued + known < 32) poll(issued - 4 + known);
      wait_steps<NP>(issued - s - 3);
      ++s;
      if (issued == s + RING - 1) {
        while (known > 0) {
          BARRIER();
          ++s;
          known = arrived(issued - 3);
          if (known < 4 && issued + 1 + known < 32) poll(issued - 3 + known);
          asm volatile("" : "+s"(known), "+s"(s));
          issue_a();
          waitv<(RING - 3) * NP>();
        }
      }
    }
    for (; s < 31; ++s) {
      BARRIER();
      wait_steps<NP>(29 - s);
    }
    BARRIER();
    waitv<0>();
  } else {
    lds_char* vp = lds + W_BYTES_STEP + half * MT * 1024 + lane * 16;
    BARRIER(); BARRIER(); BARRIER();
    unsigned vso = 4 * SLOT;
    for (int t = 4; t < 32; ++t) {
      lds_char* q = vp + vso;
      asm volatile("" : "+v"(q));
      BARRIER();
      auto complete = [&]() {
        unsigned m0 = 0xffffffffu, m1 = 0xffffffffu;
#pragma unroll
        for (int i = 0; i < NP; ++i) {
          const v4u x = *(volatile __attribute__((address_space(3))) v4u*)(q + i * 1024);
          m0 = min(min(m0, x[0]), x[1]); m1 = min(min(m1, x[2]), x[3]);
        }
        return __all(min(m0, m1) != 0);
      };
      if (__builtin_expect(!complete(), 0)) {
        const unsigned long long t0 = rtime();
        const unsigned mo = __builtin_amdgcn_readlane(uoff, t - 4), dst = ldsb + vso;
        while (!failed) {
          DMA4(" sc0 sc1", dst, va, va, rs, mo, mo, mo, mo);
          if (MT == 5) DMA1(" sc0 sc1", dst + 4096, va, rs, mo + 4096);
          waitv<0>();
          if (complete()) break;
          if (rtime() - t0 > DEADLINE_TICKS) { failed = true; lds_st32(ctl + 4, 1); wait_lgkm0(); }
        }
      }
      vso = vso == (RING - 1) * SLOT ? 0 : vso + SLOT;
    }
    BARRIER();
  }
  if (lane == 0) __hip_atomic_store(seqp, call, __ATOMIC_RELAXED, __HIP_MEMORY_SCOPE_AGENT);
  if (failed && lane == 0) __hip_atomic_store((unsigned*)(me + OFF_TMO), call, __ATOMIC_RELAXED, __HIP_MEMORY_SCOPE_AGENT);
  __builtin_amdgcn_s_waitcnt(0x0F70);
}

__device__ __forceinline__ void comm_block(const Params& P, int b, int wave, int lane TSARG) {
  const int r = P.rank;
  uint8_t* me = P.me;
  unsigned* seqp = (unsigned*)(me + OFF_SEQ) + b * NWAVE + wave;
  if (b < 28) {
    if (wave >= 4) return;
    const unsigned seq = __hip_atomic_load(seqp, __ATOMIC_RELAXED, __HIP_MEMORY_SCOPE_AGENT);
    const int d = b % 7, q = b / 7;
    const unsigned dst = (unsigned)(r + 1 + d) & 7u;
    const rsrc_t rx = mkrsrc(P.x, X_BYTES);
    const int st = grp_first_piece(d);
    const unsigned vo = (unsigned)(lane >> 3) * KLOC + (lane & 7) * 16;
    v4u a[16];
#pragma unroll
    for (int m = 0; m < 16; ++m) {
      const int n = q + 4 * wave + 16 * m;
      const int j = n >> 6, rp = (st + n) & 63;
      a[m] = __builtin_amdgcn_raw_buffer_load_b128(rx, vo, (dst * MLOC + rp * 8) * KLOC + j * 128, 0);
    }
    SCHED0();
    const unsigned call = __builtin_amdgcn_readfirstlane(seq) + 1;
    const rsrc_t rd = mkrsrc(P.peer[d] + OFF_STAGE + (call % NSLOT) * SLOT_BYTES + r * BLK_BYTES, BLK_BYTES);
#pragma unroll
    for (int m = 0; m < 16; ++m) {
      const int n = q + 4 * wave + 16 * m;
      const int j = n >> 6, rp = (st + n) & 63;
      v4u t = a[m];
#pragma unroll
      for (int c = 0; c < 4; ++c) t[c] = t[c] == 0 ? 0x80u : t[c];
      if (m >= PUSH_WIN) waitv<PUSH_WIN - 1>();
      __builtin_amdgcn_raw_buffer_store_b128(t, rd, lane * 16, j * 65536 + rp * 1024, AUX_SYS);
    }
    if (lane == 0) __hip_atomic_store(seqp, call, __ATOMIC_RELAXED, __HIP_MEMORY_SCOPE_AGENT);
  } else {
    const unsigned seq = __hip_atomic_load(seqp, __ATOMIC_RELAXED, __HIP_MEMORY_SCOPE_AGENT);
    const unsigned call = __builtin_amdgcn_readfirstlane(seq) + 1;
    const rsrc_t rz = mkrsrc(me + OFF_STAGE + ((call + 2) % NSLOT) * SLOT_BYTES, SLOT_BYTES);
    const v4u z = {0, 0, 0, 0};
    if (wave >= 8) return;
    const int u = (b - 28) * 8 + wave;
    for (int i = 0; i < 56; ++i) {
      unsigned pc = u + 32 * i;
      if (pc >= (unsigned)r * 256) pc += 256;
      __builtin_amdgcn_raw_buffer_store_b128(z, rz, lane * 16, pc * 1024, AUX_SYS);
    }
    if (lane == 0) __hip_atomic_store(seqp, call, __ATOMIC_RELAXED, __HIP_MEMORY_SCOPE_AGENT);
  }
}

__global__ void __launch_bounds__(NWAVE * 64, 3) a2a_gemm(Params P) {
  extern __shared__ __attribute__((aligned(16))) char smem[];
  lds_char* lds = (lds_char*)smem;
  const int b = blockIdx.x, lane = threadIdx.x & 63;
  const int wave = __builtin_amdgcn_readfirstlane(threadIdx.x >> 6);
  if (b < 32) {
    comm_block(P, b, wave, lane TSPASS);
  } else {
    const int cb = b - 32, g = cb >> 5, nt = (cb & 7) * 4 + ((cb >> 3) & 3);
    if (wave >= 8) {
      wload_wave(P, lds, g, nt, wave - 8, lane TSPASS);
    } else if (g < 4) {
      if (wave < 4) compute_wave<5>(P, lds, g, nt, wave, lane TSPASS); else aload_wave<5>(P, lds, g, wave - 4, lane, b TSPASS);
    } else {
      if (wave < 4) compute_wave<4>(P, lds, g, nt, wave, lane TSPASS); else aload_wave<4>(P, lds, g, wave - 4, lane, b TSPASS);
    }
  }
}

}

extern "C" void kernel_launch(int rank, int nranks, ncclComm_t comm, hipStream_t stream,
                              void* const* d_in, const int* in_sizes, int n_in,
                              void* d_out, int out_size, const int* in_dtypes, int out_dtype,
                              void* const* d_out_all, void* const* d_ws_all, size_t ws_bytes) {
  if (nranks != NR || n_in < 4 || ws_bytes < WS_USED || (unsigned)rank >= (unsigned)NR ||
      in_sizes[0] != (int)X_BYTES || in_sizes[1] != (int)W_BYTES || out_size != MLOC * NDIM) {
    fprintf(stderr, "unexpected problem shape\n");
    return;
  }
  HIP_CHECK(hipFuncSetAttribute((const void*)a2a_gemm, hipFuncAttributeMaxDynamicSharedMemorySize, LDS_BYTES));
  Params P;
  P.x = (const uint8_t*)d_in[0]; P.w = (const uint8_t*)d_in[1];
  P.sx = (const float*)d_in[2]; P.sw = (const float*)d_in[3];
  P.out = (float*)d_out;
  P.me = (uint8_t*)d_ws_all[rank];
  for (int d = 0; d < NR; ++d) P.peer[d] = (uint8_t*)d_ws_all[(rank + 1 + d) % NR];
  P.rank = rank;
  hipLaunchKernelGGL(a2a_gemm, dim3(256), dim3(NWAVE * 64), LDS_BYTES, stream, P);
  HIP_CHECK(hipGetLastError());
}
```

```cpp
#include <hip/hip_runtime.h>
#include <rccl/rccl.h>
#include <cstdio>
#include <cstdint>

#define HIP_CHECK(expr) do { hipError_t _e = (expr); if (_e != hipSuccess) { \
    fprintf(stderr, "HIP error %s at %s:%d\n", hipGetErrorString(_e), __FILE__, __LINE__); \
    return; } } while (0)

namespace {

typedef int v2i __attribute__((ext_vector_type(2)));
typedef unsigned v4u __attribute__((ext_vector_type(4)));
typedef int v8i __attribute__((ext_vector_type(8)));
typedef float v4f __attribute__((ext_vector_type(4)));
typedef __attribute__((address_space(3))) char lds_char;
typedef __amdgpu_buffer_rsrc_t rsrc_t;

constexpr int NWAVE = 12;
constexpr int NR = 8;
constexpr int KDIM = 4096, NDIM = 4096, MLOC = 512, KLOC = 512;
constexpr unsigned X_BYTES = 4096u * KLOC;
constexpr unsigned W_BYTES = (unsigned)KDIM * NDIM;
constexpr unsigned OUT_BYTES = (unsigned)MLOC * NDIM * 4u;
constexpr unsigned BLK_BYTES = MLOC * KLOC;
constexpr unsigned SLOT_BYTES = NR * BLK_BYTES;
constexpr unsigned NSLOT = 3;
constexpr unsigned OFF_SEQ = 0;
constexpr unsigned OFF_TMO = 16384;
constexpr unsigned OFF_STAGE = 65536;
constexpr unsigned STAGE_BYTES = NSLOT * SLOT_BYTES;
constexpr unsigned WS_USED = OFF_STAGE + STAGE_BYTES;
constexpr unsigned long long DEADLINE_TICKS = 30000000ull;

constexpr int RING = 6;
constexpr int W_BYTES_STEP = 16384, A_BYTES_STEP = 10240, SLOT = W_BYTES_STEP + A_BYTES_STEP;
constexpr int LDS_CTL = RING * SLOT;
constexpr int LDS_BYTES = LDS_CTL + 64;
constexpr int AUX_SC1 = 16, AUX_SYS = 17;

#define PROF_GAP 2000000u
#define PROF_EVERY 0
#define PROF_S 7
#define LOAD_PRIO 1
#define POLL_WIN 6
#define PUSH_MODE 0
#define PUSH_T0 40
#define PUSH_TAU 120
#define PUSH_SLEEP 4
#define HINT_BACK 0

struct KArgs {
  const uint8_t* x; const uint8_t* w; uint8_t* me; float* out; const float* sx; const float* sw;
  int rank; int pad[3];
  uint8_t* peer[NR];
};
struct Params {
  const uint8_t* x; const uint8_t* w; uint8_t* me; float* out; const float* sx; const float* sw;
  uint8_t* peer;
  int rank;
};

#define STAMP(i)
#define TSARG
#define TSPASS
template <int N> __device__ __forceinline__ void waitv() { asm volatile("s_waitcnt vmcnt(%0)" :: "i"(N) : "memory"); }
__device__ __forceinline__ void wait_lgkm0() { asm volatile("" ::: "memory"); __builtin_amdgcn_s_waitcnt(0xC07F); asm volatile("" ::: "memory"); }
__device__ __forceinline__ unsigned long long rtime() { return __builtin_amdgcn_s_memrealtime(); }
__device__ __forceinline__ rsrc_t mkrsrc(const void* p, unsigned bytes) {
  return __builtin_amdgcn_make_buffer_rsrc((void*)p, (short)0, (int)bytes, 0x00020000);
}
__device__ __forceinline__ unsigned ctl_ld(rsrc_t rc, unsigned off) { return __builtin_amdgcn_raw_buffer_load_b32(rc, 0, off, AUX_SC1); }
__device__ __forceinline__ void ctl_st(rsrc_t rc, unsigned off, unsigned v, int lane) {
  __builtin_amdgcn_raw_buffer_store_b32(v, rc, lane == 0 ? 0u : 0x80000000u, off, AUX_SC1);
}
__device__ __forceinline__ void st16(v4u v, rsrc_t rs, unsigned voff, unsigned soff) {
  __builtin_amdgcn_sched_barrier(0);
  __builtin_amdgcn_raw_buffer_store_b128(v, rs, voff, soff, AUX_SYS);
  __builtin_amdgcn_sched_barrier(0);
  asm volatile("s_nop 2" :: "v"(v) : "memory");
  __builtin_amdgcn_sched_barrier(0);
}
__device__ __forceinline__ float ld_const(const float* p) { return *(const __attribute__((address_space(4))) float*)p; }
__device__ __forceinline__ int grp_first_piece(int g) { return 8 * g + 2 * min(g, 4); }

__device__ __forceinline__ void step_src(int s, int g, int r, int& p, int& j, int& d) {
  const int i = s - 4, jr = (i * 37) >> 8, dr = g - (i - 7 * jr);
  d = dr < 0 ? dr + 7 : dr;
  j = s < 4 ? s : jr;
  p = s < 4 ? r : (r - 1 - d) & 7;
}

__device__ __forceinline__ unsigned lds_ld32(lds_char* p) { return *(volatile __attribute__((address_space(3))) unsigned*)p; }
__device__ __forceinline__ void lds_st32(lds_char* p, unsigned v) { *(volatile __attribute__((address_space(3))) unsigned*)p = v; }

#define SCHED0() __builtin_amdgcn_sched_barrier(0)
#define BARRIER() do { asm volatile("" ::: "memory"); __builtin_amdgcn_s_barrier(); asm volatile("" ::: "memory"); } while (0)
template <int MT> struct Frags { v4u a[MT][2]; v8i b[2]; };

__device__ __forceinline__ void mma1(const v4u (&a)[2], const v8i& bb, v4f& acc) {
  const v8i aa = v8i{(int)a[0][0], (int)a[0][1], (int)a[0][2], (int)a[0][3], (int)a[1][0], (int)a[1][1], (int)a[1][2], (int)a[1][3]};
  acc = __builtin_amdgcn_mfma_scale_f32_16x16x128_f8f6f4(aa, bb, acc, 0, 1, 0, 0, 0, 0);
  asm volatile("" : "+v"(acc));
}

template <int MT>
__device__ __forceinline__ void compute_wave(const Params& P, lds_char* lds, int g, int nt, int w, int lane TSARG) {
  const int gq = lane >> 4, l15 = lane & 15;
  const int q = l15 >> 1;
  const int fsw = ((q >> 1) & 3) | ((gq & 1) << 2);
  const int boff0 = (16 * gq + q) * 128 + (((2 * w) ^ fsw) << 4) + 8 * (lane & 1);
  const int aoff0 = W_BYTES_STEP + l15 * 128 + ((gq ^ (l15 & 7)) << 4);
  lds_char* ctl = lds + LDS_CTL;

  v4f acc[MT][2];
#pragma unroll
  for (int m = 0; m < MT; ++m) { acc[m][0] = v4f{0, 0, 0, 0}; acc[m][1] = v4f{0, 0, 0, 0}; }

  int ob0 = boff0, ob1 = boff0 ^ 16, oa0 = aoff0, oa1 = aoff0 ^ 64;
  asm volatile("" : "+v"(ob0), "+v"(ob1), "+v"(oa0), "+v"(oa1));
  lds_char* pb[2]; lds_char* pa[2];
  int so = 0;
  auto set_addr = [&]() {
    pb[0] = lds + (ob0 + so); pb[1] = lds + (ob1 + so);
    pa[0] = lds + (oa0 + so); pa[1] = lds + (oa1 + so);
    asm volatile("" : "+v"(pb[0]), "+v"(pb[1]), "+v"(pa[0]), "+v"(pa[1]));
  };
  auto read_b = [&](Frags<MT>& f, int t) {
    v2i r[4];
#pragma unroll
    for (int u = 0; u < 4; ++u)
      r[u] = __builtin_amdgcn_ds_read_tr8_b64_v2i32((__attribute__((address_space(3))) v2i*)(pb[t] + (u >> 1) * 8192 + (u & 1) * 1024));
    f.b[t] = v8i{r[0][0], r[0][1], r[1][0], r[1][1], r[2][0], r[2][1], r[3][0], r[3][1]};
  };
  auto read_a = [&](Frags<MT>& f, int m) {
    f.a[m][0] = *(__attribute__((address_space(3))) v4u*)(pa[0] + m * 2048);
    f.a[m][1] = *(__attribute__((address_space(3))) v4u*)(pa[1] + m * 2048);
  };
  auto step = [&](Frags<MT>& fn, const Frags<MT>& fp) {
#pragma unroll
    for (int i = 0; i < 2 * MT; ++i) {
      mma1(fp.a[i >> 1], fp.b[i & 1], acc[i >> 1][i & 1]);
      if (i == 0) { so = so == (RING - 1) * SLOT ? 0 : so + SLOT; set_addr(); }
      if (i == 1) { wait_lgkm0(); BARRIER(); read_a(fn, 0); }
      if (i == 2) read_b(fn, 0);
      if (i == 3) read_b(fn, 1);
      if (i >= 4 && i < MT + 3) read_a(fn, i - 3);
      SCHED0();
    }
  };

  Frags<MT> f0, f1;
  set_addr();
  BARRIER();
  STAMP(2);
  read_a(f0, 0); read_b(f0, 0); read_b(f0, 1);
#pragma unroll
  for (int m = 1; m < MT; ++m) read_a(f0, m);
  SCHED0();
  for (int s = 1; s < 31; s += 2) {
    step(f1, f0);
    step(f0, f1);
  }
  step(f1, f0);
  STAMP(10);
  const unsigned fail = lds_ld32(ctl + 4);
  const float scale0 = ld_const(P.sx) * ld_const(P.sw);
#pragma unroll
  for (int i = 0; i < 2 * MT; ++i) mma1(f1.a[i >> 1], f1.b[i & 1], acc[i >> 1][i & 1]);

  const float scale = fail != 0 ? __builtin_nanf("") : scale0;
  STAMP(11);
  const rsrc_t ro = mkrsrc(P.out, OUT_BYTES);
  const unsigned ov = (unsigned)(grp_first_piece(g) * 8 + 4 * gq) * (NDIM * 4) + (unsigned)(nt * 128 + 32 * w + l15) * 4;
#pragma unroll
  for (int m = 0; m < MT; ++m)
#pragma unroll
    for (int j = 0; j < 4; ++j)
#pragma unroll
      for (int t = 0; t < 2; ++t)
        __builtin_amdgcn_raw_buffer_store_b32(__builtin_bit_cast(unsigned, acc[m][t][j] * scale), ro, ov + t * 64,
                                              (m * 16 + j) * (NDIM * 4), AUX_SC1);
}

template <int PER> __device__ __forceinline__ void wait_steps(int n) {
  waitv<4 * PER>();
  if (n < 4) { waitv<3 * PER>(); if (n < 3) { waitv<2 * PER>(); if (n < 2) { waitv<PER>(); if (n < 1) waitv<0>(); } } }
}

#define DMA4(POL, dst, v0, v1, rs, s0, s1, s2, s3) \
  asm volatile("s_mov_b32 m0, %0\n\ts_nop 3\n\t" \
               "buffer_load_dwordx4 %1, %3, %4 offen" POL " lds\n\t" \
               "buffer_load_dwordx4 %1, %3, %5 offen offset:1024" POL " lds\n\t" \
               "buffer_load_dwordx4 %2, %3, %6 offen offset:2048" POL " lds\n\t" \
               "buffer_load_dwordx4 %2, %3, %7 offen offset:3072" POL " lds" \
               :: "s"(dst), "v"(v0), "v"(v1), "s"(rs), "s"(s0), "s"(s1), "s"(s2), "s"(s3) : "memory")

__device__ __forceinline__ void wload_wave(const Params& P, lds_char* lds, int g, int nt, int u, int lane TSARG) {
  const rsrc_t rw = mkrsrc(P.w, W_BYTES);
  const int r = P.rank;
  const unsigned wv0 = (unsigned)(32 * u + (lane >> 3)) * NDIM + (unsigned)(((lane & 7) ^ ((lane >> 4) & 3)) << 4) + nt * 128;
  const unsigned wv1 = wv0 ^ 64u;
  const unsigned ldsb = (unsigned)(uintptr_t)lds + u * 4096;
  unsigned so = 0;
  unsigned koff;
  { int p, j, d; step_src(lane & 31, g, r, p, j, d); koff = (unsigned)(p * KLOC + j * 128) * NDIM; }
  constexpr unsigned ST = 8 * NDIM - 1024;
  unsigned o0, o1, o2, o3, dst;
  auto form = [&](int s) {
    o0 = s < 4 ? (unsigned)(r * KLOC + s * 128) * NDIM : __builtin_amdgcn_readlane(koff, s);
    o1 = o0 + ST; o2 = o0 + 2 * ST; o3 = o0 + 3 * ST;
    dst = ldsb + so;
    so = so == (RING - 1) * SLOT ? 0 : so + SLOT;
    asm volatile("" : "+s"(o0), "+s"(o1), "+s"(o2), "+s"(o3), "+s"(dst), "+s"(so));
  };
  auto issue_w = [&](int next) {
    DMA4("", dst, wv0, wv1, rw, o0, o1, o2, o3);
    form(next);
  };
  __builtin_amdgcn_s_setprio(LOAD_PRIO);
  form(0);
#pragma unroll
  for (int s = 0; s < RING - 1; ++s) issue_w(s + 1);
  waitv<4 * (RING - 2)>();
  STAMP(1);
  for (int s = 0; s < 32 - (RING - 1); ++s) {
    BARRIER();
    issue_w((s + RING) & 31);
    waitv<4 * (RING - 2)>();
  }
  BARRIER(); waitv<12>();
  BARRIER(); waitv<8>();
  BARRIER(); waitv<4>();
  BARRIER(); waitv<0>();
  BARRIER();
  STAMP(2);
}

#define DMA1(POL, dst, v0, rs, s0) \
  asm volatile("s_mov_b32 m0, %0\n\ts_nop 3\n\t" \
               "buffer_load_dwordx4 %1, %2, %3 offen" POL " lds" \
               :: "s"(dst), "v"(v0), "s"(rs), "s"(s0) : "memory")

template <int MT>
__device__ __forceinline__ void aload_wave(const Params& P, lds_char* lds, int g, int aw, int lane, int b TSARG) {
  constexpr int NP = MT;
  const int role = aw >> 1, half = aw & 1;
  const int r = P.rank;
  uint8_t* me = P.me;
  const rsrc_t rs = mkrsrc(me + OFF_STAGE, STAGE_BYTES);
  const int p0 = grp_first_piece(g) + half * MT;
  const rsrc_t rc = mkrsrc(me, OFF_STAGE);
  const unsigned seqo = OFF_SEQ + (b * NWAVE + 4 + aw) * 4;
  unsigned seqv;
  asm volatile("s_nop 4\n\tbuffer_load_dword %0, off, %1, %2 sc1" : "=v"(seqv) : "s"(rc), "s"(seqo) : "memory");
  const unsigned sw = (unsigned)(((lane & 7) ^ (lane >> 3)) << 4);
  const unsigned ldsb = (unsigned)(uintptr_t)lds + W_BYTES_STEP + half * MT * 1024;
  lds_char* ctl = lds + LDS_CTL;
  __builtin_amdgcn_s_setprio(LOAD_PRIO);
  if (role == 0) {
    const rsrc_t rx = mkrsrc(P.x, X_BYTES);
    const unsigned vx = (unsigned)(lane >> 3) * KLOC + sw;
    constexpr unsigned ST = 8 * KLOC - 1024;
#pragma unroll
    for (int s = 0; s < 4; ++s) {
      const unsigned mo = (unsigned)(r * MLOC + p0 * 8) * KLOC + s * 128, dst = ldsb + s * SLOT;
      DMA4("", dst, vx, vx, rx, mo, mo + ST, mo + 2 * ST, mo + 3 * ST);
      if (MT == 5) DMA1("", dst + 4096, vx, rx, mo + 32 * KLOC);
    }
  } else {
    if (half == 0 && lane == 0) lds_st32(ctl + 4, 0);
    wait_lgkm0();
  }
  asm volatile("s_waitcnt vmcnt(%1)" : "+v"(seqv) : "i"(4 * NP) : "memory");
  if (role == 1) asm volatile("s_waitcnt vmcnt(0)" : "+v"(seqv) :: "memory");
  const unsigned call = __builtin_amdgcn_readfirstlane(seqv) + 1;
  const unsigned slot_off = (call % NSLOT) * SLOT_BYTES;
  unsigned hoff, uoff;
  {
    int p, j, d; step_src(min(lane, 27) + 4, g, r, p, j, d);
    const int fd = grp_first_piece(d);
    const int n = max(j * 64 + ((p0 + NP - 1 - fd) & 63) - HINT_BACK, 0);
    hoff = slot_off + p * BLK_BYTES + (n >> 6) * 65536 + ((fd + n) & 63) * 1024 + 1020;
    uoff = slot_off + p * BLK_BYTES + j * 65536 + p0 * 1024;
  }
  const unsigned va = (unsigned)(lane >> 3) * 128 + sw;
  bool failed = false;
  if (role == 0) {
    asm volatile("v_mov_b32 v167, 0" ::: "v167");
    auto poll = [&](int first) {
      const unsigned long long m = (((1ull << POLL_WIN) - 1) << first) & 0xfffffffull;
      asm volatile("s_mov_b64 exec, %0\n\tbuffer_load_dword v167, %1, %2, 0 offen sc0 sc1\n\ts_mov_b64 exec, -1"
                   :: "s"(m), "v"(hoff), "s"(rs) : "memory", "v167");
    };
    auto arrived = [&](int first) {
      unsigned hv;
      asm volatile("v_mov_b32 %0, v167" : "=v"(hv) :: "v167");
      const unsigned m = (unsigned)__ballot(hv != 0);
      return __builtin_ctz(~(m >> first));
    };
    int issued = 4, known = 0, s = 0;
    unsigned so = 4 * SLOT, mo, dst;
    auto form = [&]() {
      mo = __builtin_amdgcn_readlane(uoff, (issued - 4) & 31);
      dst = ldsb + so;
      so = so == (RING - 1) * SLOT ? 0 : so + SLOT;
      asm volatile("" : "+s"(mo), "+s"(dst), "+s"(so));
    };
    auto issue_a = [&]() {
      form();
      ++issued;
      unsigned m1 = mo + 4096, d1 = dst + 4096;
      asm volatile("" : "+s"(m1), "+s"(d1), "+s"(issued));
      DMA4(" sc0 sc1", dst, va, va, rs, mo, mo, mo, mo);
      if (MT == 5) DMA1(" sc0 sc1", d1, va, rs, m1);
    };
    poll(0);
    waitv<2 * NP>();
    STAMP(1);
    while (issued < 32) {
      BARRIER();
      known = arrived(issued - 4);
      if (known == 0 && issued <= s + 2) {
        const unsigned long long t0 = rtime();
        while (!failed) {
          poll(issued - 4);
          waitv<0>();
          known = arrived(issued - 4);
          if (known) break;
          if (rtime() - t0 > DEADLINE_TICKS) failed = true;
        }
        if (failed) { lds_st32(ctl + 4, 1); wait_lgkm0(); known = 1; }
      }
      const int k = min(known, min(s + RING, 32) - issued);
      for (int i = 0; i < k; ++i) issue_a();
      known -= k;
      if (known < 4 && issued + known < 32) poll(issued - 4 + known);
      wait_steps<NP>(issued - s - 3);
      ++s;
      if (issued == s + RING - 1) {
        while (known > 0) {
          BARRIER();
          ++s;
          known = arrived(issued - 3);
          if (known < 4 && issued + 1 + known < 32) poll(issued - 3 + known);
          asm volatile("" : "+s"(known), "+s"(s));
          issue_a();
          waitv<(RING - 3) * NP>();
        }
      }
    }
    for (; s < 31; ++s) {
      BARRIER();
      wait_steps<NP>(29 - s);
    }
    BARRIER();
    waitv<0>();
  } else {
    lds_char* vp = lds + W_BYTES_STEP + half * MT * 1024 + lane * 16;
    BARRIER(); BARRIER(); BARRIER();
    unsigned vso = 4 * SLOT;
    for (int t = 4; t < 32; ++t) {
      lds_char* q = vp + vso;
      asm volatile("" : "+v"(q));
      BARRIER();
      auto complete = [&]() {
        unsigned m0 = 0xffffffffu, m1 = 0xffffffffu;
#pragma unroll
        for (int i = 0; i < NP; ++i) {
          const v4u x = *(volatile __attribute__((address_space(3))) v4u*)(q + i * 1024);
          m0 = min(min(m0, x[0]), x[1]); m1 = min(min(m1, x[2]), x[3]);
        }
        return __all(min(m0, m1) != 0);
      };
      if (__builtin_expect(!complete(), 0)) {
        const unsigned long long t0 = rtime();
        const unsigned mo = __builtin_amdgcn_readlane(uoff, t - 4), dst = ldsb + vso;
        while (!failed) {
          DMA4(" sc0 sc1", dst, va, va, rs, mo, mo, mo, mo);
          if (MT == 5) DMA1(" sc0 sc1", dst + 4096, va, rs, mo + 4096);
          waitv<0>();
          if (complete()) break;
          if (rtime() - t0 > DEADLINE_TICKS) { failed = true; lds_st32(ctl + 4, 1); wait_lgkm0(); }
        }
      }
      vso = vso == (RING - 1) * SLOT ? 0 : vso + SLOT;
    }
    BARRIER();
  }
  ctl_st(rc, seqo, call, lane);
  if (failed) ctl_st(rc, OFF_TMO, call, lane);
  __builtin_amdgcn_s_waitcnt(0x0F70);
}

__device__ __forceinline__ void comm_block(const Params& P, int b, int wave, int lane, unsigned tstart TSARG) {
  const int r = P.rank;
  uint8_t* me = P.me;
  const rsrc_t rc = mkrsrc(me, OFF_STAGE);
  const unsigned seqo = OFF_SEQ + (b * NWAVE + wave) * 4;
  if (__builtin_expect(b < 28, 1)) {
    if (wave >= 8) return;
    const int q = (b * 37) >> 8, d = b - 7 * q;
    uint8_t* peer = P.peer;
    const rsrc_t rx = mkrsrc(P.x, X_BYTES);
    STAMP(6);
    const unsigned seq = ctl_ld(rc, seqo);
    const unsigned dst = (unsigned)(r + 1 + d) & 7u;
    const unsigned rp0 = (unsigned)(grp_first_piece(d) + q + 4 * wave) & 63u;
    const unsigned vo = (unsigned)(lane >> 3) * KLOC + (lane & 7) * 16;
    const unsigned xo0 = (dst * MLOC + rp0 * 8) * KLOC, xo1 = (dst * MLOC + (rp0 ^ 32u) * 8) * KLOC;
    v4u a[8];
    auto ldx = [&](int m) { a[m] = __builtin_amdgcn_raw_buffer_load_b128(rx, vo + (m >> 1) * 128, (m & 1) ? xo1 : xo0, 0); SCHED0(); };
    ldx(0); ldx(1);
    STAMP(7);
    const unsigned call = __builtin_amdgcn_readfirstlane(seq) + 1;
    STAMP(8);
    const rsrc_t rd = mkrsrc(peer + OFF_STAGE + (call % NSLOT) * SLOT_BYTES + r * BLK_BYTES, BLK_BYTES);
#pragma unroll
    for (int m = 0; m < 8; ++m) {
      v4u t = a[m];
#pragma unroll
      for (int c = 0; c < 4; ++c) t[c] = t[c] == 0 ? 0x80u : t[c];
      asm volatile("" : "+v"(t));
      st16(t, rd, lane * 16, (m >> 1) * 65536 + (rp0 ^ ((m & 1) * 32u)) * 1024);
      if (m + 2 < 8) ldx(m + 2);
    }
    ctl_st(rc, seqo, call, lane);
  } else {
    const unsigned seq = ctl_ld(rc, seqo);
    const unsigned call = __builtin_amdgcn_readfirstlane(seq) + 1;
    const rsrc_t rz = mkrsrc(me + OFF_STAGE + ((call + 2) % NSLOT) * SLOT_BYTES, SLOT_BYTES);
    const v4u z = {0, 0, 0, 0};
    if (wave >= 8) return;
    const int u = (b - 28) * 8 + wave;
#pragma unroll 2
    for (int i = 0; i < 56; ++i) {
      unsigned pc = u + 32 * i;
      if (pc >= (unsigned)r * 256) pc += 256;
      st16(z, rz, lane * 16, pc * 1024);
    }
    ctl_st(rc, seqo, call, lane);
  }
}

__global__ void __launch_bounds__(NWAVE * 64, 3) a2a_gemm(KArgs K) {
  extern __shared__ __attribute__((aligned(16))) char smem[];
  lds_char* lds = (lds_char*)smem;
  const int b = blockIdx.x, lane = threadIdx.x & 63;
  const int wave = __builtin_amdgcn_readfirstlane(threadIdx.x >> 6);
  const unsigned tstart = (unsigned)rtime();
  Params P;
  P.x = K.x; P.w = K.w; P.me = K.me; P.out = K.out; P.sx = K.sx; P.sw = K.sw; P.rank = K.rank;
  P.peer = K.peer[(b - 7 * ((b * 37) >> 8)) & 7];
  asm volatile("" : "+s"(P.x), "+s"(P.w), "+s"(P.me), "+s"(P.out), "+s"(P.sx), "+s"(P.sw), "+s"(P.rank), "+s"(P.peer));
  if (__builtin_expect(b < 32, 1)) {
    comm_block(P, b, wave, lane, tstart TSPASS);
  } else {
    const int cb = b - 32, g = cb >> 5, nt = (cb & 7) * 4 + ((cb >> 3) & 3);
    if (wave >= 8) {
      wload_wave(P, lds, g, nt, wave - 8, lane TSPASS);
    } else if (g < 4) {
      if (wave < 4) compute_wave<5>(P, lds, g, nt, wave, lane TSPASS); else aload_wave<5>(P, lds, g, wave - 4, lane, b TSPASS);
    } else {
      if (wave < 4) compute_wave<4>(P, lds, g, nt, wave, lane TSPASS); else aload_wave<4>(P, lds, g, wave - 4, lane, b TSPASS);
    }
  }
}

}

extern "C" void kernel_launch(int rank, int nranks, ncclComm_t comm, hipStream_t stream,
                              void* const* d_in, const int* in_sizes, int n_in,
                              void* d_out, int out_size, const int* in_dtypes, int out_dtype,
                              void* const* d_out_all, void* const* d_ws_all, size_t ws_bytes) {
  if (nranks != NR || n_in < 4 || ws_bytes < WS_USED || (unsigned)rank >= (unsigned)NR ||
      in_sizes[0] != (int)X_BYTES || in_sizes[1] != (int)W_BYTES || out_size != MLOC * NDIM) {
    fprintf(stderr, "unexpected problem shape\n");
    return;
  }
  HIP_CHECK(hipFuncSetAttribute((const void*)a2a_gemm, hipFuncAttributeMaxDynamicSharedMemorySize, LDS_BYTES));
  KArgs P;
  P.x = (const uint8_t*)d_in[0]; P.w = (const uint8_t*)d_in[1];
  P.sx = (const float*)d_in[2]; P.sw = (const float*)d_in[3];
  P.out = (float*)d_out;
  P.me = (uint8_t*)d_ws_all[rank];
  for (int d = 0; d < NR; ++d) P.peer[d] = (uint8_t*)d_ws_all[(rank + 1 + d) % NR];
  P.rank = rank;
  hipLaunchKernelGGL(a2a_gemm, dim3(256), dim3(NWAVE * 64), LDS_BYTES, stream, P);
  HIP_CHECK(hipGetLastError());
}
```
